```python
import numpy as np

kernel_name = "gpu_ag_gemm_m4096_k4096_n4096_bf16_reluqe4m3"


ABSMAX_RTOL = 0.09

COMPUTE_DTYPE = "bfloat16"
EPILOGUE = "relu_quant_e4m3"
IN_QUANT = None
OUT_QUANT = "e4m3"
IN_DTYPES = {"x": "bfloat16", "w_mat": "bfloat16"}
WORLD_SIZE = 8

def _to_bf16(x):
    u = np.ascontiguousarray(x, dtype=np.float32).view(np.uint32)
    return ((u + 0x7FFF + ((u >> 16) & 1)) & 0xFFFF0000).view(np.float32)

def setup_inputs():

    x = _to_bf16(np.random.randn(4096, 4096).astype(np.float32))
    w_mat = _to_bf16((np.random.randn(4096, 4096) / np.sqrt(4096)).astype(np.float32))
    return {"x": x, "w_mat": w_mat}

def _snap_fp8(x, mant_bits, max_abs, min_normal):
    a = np.abs(x.astype(np.float64))
    _m, _e = np.frexp(a)
    q = np.where(a >= min_normal, np.ldexp(1.0, _e - 1 - mant_bits),
                 min_normal * 2.0 ** -mant_bits)
    snapped = np.minimum(np.round(a / q) * q, max_abs)
    return (np.sign(x) * snapped).astype(np.float32)
def _quant_dequant(y, amax):
    scale = amax / 448.0
    q = _snap_fp8(y / scale, 3, 448.0, 2.0 ** -6)
    return (q * scale).astype(np.float32)
def _epilogue(y):
    y = np.maximum(y, 0.0)
    return _quant_dequant(y, np.abs(y).max())

def reference(x, w_mat):
    y = (x @ w_mat).astype(np.float32)
    return _epilogue(y)
```

```cpp
#include <hip/hip_runtime.h>
#include <rccl/rccl.h>
#include <cstdio>
#include <cstdint>

#define HIP_CHECK(expr) do { hipError_t _e = (expr); if (_e != hipSuccess) { \
    fprintf(stderr, "HIP error %s at %s:%d\n", hipGetErrorString(_e), __FILE__, __LINE__); \
    return; } } while (0)

typedef short bf16x8 __attribute__((ext_vector_type(8)));
typedef float f32x4 __attribute__((ext_vector_type(4)));
typedef unsigned u32x4 __attribute__((ext_vector_type(4)));
typedef unsigned long long u64;
typedef const u32x4 __attribute__((address_space(1)))* gcptr;

constexpr int NR = 8;
constexpr int KD = 4096;
constexpr int NS = 512;
constexpr int KSTEPS = 64;
constexpr int TA = 5, TB = 3;
#define PACE 16000
constexpr int NWG = 256;
#define WPLV 36
constexpr int WPL = WPLV;
constexpr int NT = 320;
constexpr unsigned long long DEADLINE = 40000000ull;

constexpr size_t O_SEQ = 0, O_TMO = 128, O_EXIT = 256, O_GCNT = 512, O_GMAX = 2048, O_OCNT = 4096,
                 O_AMAX = 8192, O_DONE = 12288,
                 O_FLAGX = 65536,
                 O_FLAGW = 131072;
constexpr size_t O_XG = 1ull << 20;
constexpr size_t XG_SRC = (size_t)TA * KSTEPS * 8192;
constexpr size_t O_WT = O_XG + NR * XG_SRC;
constexpr size_t WT_SLOT = (size_t)KSTEPS * 16384;
constexpr size_t WS_NEED = O_WT + 8 * WT_SLOT;

constexpr int NSTAGE = 5;
constexpr int STAGE_BYTES = 24576;
constexpr int LDS_BYTES = NSTAGE * STAGE_BYTES + 1024;

#define RLX __ATOMIC_RELAXED
#define SYS __HIP_MEMORY_SCOPE_SYSTEM
#define AGT __HIP_MEMORY_SCOPE_AGENT

__device__ __forceinline__ bool wait_flag(unsigned* flag, unsigned seq, unsigned* tmo) {
    unsigned long long t0 = __builtin_amdgcn_s_memrealtime();
    while (__hip_atomic_load(flag, RLX, SYS) < seq) {
        if (__hip_atomic_load(tmo, RLX, AGT) == seq || __builtin_amdgcn_s_memrealtime() - t0 > DEADLINE) {
            __hip_atomic_store(tmo, seq, RLX, AGT);
            return false;
        }
        __builtin_amdgcn_s_sleep(1);
    }
    return true;
}

#define STAMP(i) do {} while (0)
__device__ __forceinline__ void drain() { asm volatile("s_waitcnt vmcnt(0)" ::: "memory"); }
__device__ __forceinline__ void store_wt(void* p, u32x4 v) {
    asm volatile("global_store_dwordx4 %0, %1, off sc0 sc1\n\ts_nop 1" :: "v"(p), "v"(v) : "memory");
}
__device__ __forceinline__ void store_flag(unsigned* p, unsigned v) {
    asm volatile("global_store_dword %0, %1, off sc0 sc1" :: "v"(p), "v"(v) : "memory");
}

struct Tile { unsigned sx; int tx, grow, ct; };
__device__ __forceinline__ Tile tile_of(int id, int rank) {
    Tile t; const int rt = id >> 3; t.ct = id & 7;
    if (rt < 4 * TA) { t.sx = (unsigned)(rank + 2 * (rt / TA)) % NR; t.tx = rt % TA; t.grow = t.sx * NS + t.tx * 64; }
    else { const int r2 = rt - 4 * TA; t.sx = (unsigned)(rank + 2 * (r2 / TB) + 1) % NR; t.tx = r2 % TB; t.grow = t.sx * NS + TA * 64 + t.tx * 64; }
    return t;
}

__global__ void __launch_bounds__(NT, 2)
ag_gemm_fused(const unsigned short* __restrict__ x, const unsigned short* __restrict__ w, int rank,
              void* const* __restrict__ out_all, void* const* __restrict__ ws_all) {
    __shared__ __attribute__((aligned(16))) char lds[LDS_BYTES];
    char* ws = (char*)ws_all[rank];
    const int tid = threadIdx.x, lane = tid & 63, wv = tid >> 6;
    unsigned* tmo = (unsigned*)(ws + O_TMO);
    const unsigned seq = __hip_atomic_load((unsigned*)(ws + O_SEQ), RLX, AGT) + 1u;
    const int g = blockIdx.x;
    typedef __attribute__((address_space(3))) unsigned lds_u32;
    lds_u32* lsync = (lds_u32*)(__attribute__((address_space(3))) char*)(lds + NSTAGE * STAGE_BYTES + 128);
    if (tid == 0) { lsync[0] = 0u; lsync[1] = 0u; }
    __syncthreads();
    STAMP(0);

    if (wv == 4) {
        for (int u = g; u < TA * KSTEPS; u += NWG) {
            const int ks = u / TA, t = u % TA;
            const unsigned short* src = x + (size_t)(t * 64 + (lane & 15)) * KD + ks * 64 + (lane >> 4) * 8;
            u32x4 v[8];
#pragma unroll
            for (int f = 0; f < 8; ++f) v[f] = *(const u32x4*)(src + (size_t)(f >> 1) * 16 * KD + (f & 1) * 32);
            char* dst = ws + O_XG + (size_t)rank * XG_SRC + ((size_t)t * KSTEPS + ks) * 8192 + lane * 16;
#pragma unroll
            for (int f = 0; f < 8; ++f) store_wt(dst + f * 1024, v[f]);
            drain();
            store_flag((unsigned*)(ws + O_FLAGX) + (rank * TA + t) * KSTEPS + ks, seq);
        }
        if (g < 7 * WPL) {
            const int o = 1 + g % 7, u0 = g / 7;
            const unsigned d = (unsigned)(rank + o) % NR;
            const bool odd = o & 1;
            const int j = o >> 1;
            const int rbase = odd ? TA * 64 : 0;
            char* dws = (char*)ws_all[d];
            char* dxg = dws + O_XG + (size_t)rank * XG_SRC;
            char* dwt = dws + O_WT + (size_t)(4 + j) * WT_SLOT;
            const char* lwt = ws + O_WT + (size_t)j * WT_SLOT;
            unsigned* dfl = (unsigned*)(dws + O_FLAGX) + rank * TA * KSTEPS;
            u64* dfw = (u64*)(dws + O_FLAGW) + (4 + j) * KSTEPS * 2;
            u32x4 va[8], vb[8];
            auto xload = [&](u32x4 (&v)[8], int u) {
                const int ks = u / 5, t = u % 5;
                const bool isw = odd && t >= TB;
                if (isw) {
                    const u64* lf = (const u64*)(ws + O_FLAGW) + (j * KSTEPS + ks) * 2 + (t - TB);
                    unsigned long long t0 = __builtin_amdgcn_s_memrealtime();
                    for (;;) {
                        const u64 f = __hip_atomic_load(lf, RLX, AGT);
                        if ((unsigned)f >= seq && (unsigned)(f >> 32) >= seq) break;
                        if (__hip_atomic_load(tmo, RLX, AGT) == seq || __builtin_amdgcn_s_memrealtime() - t0 > DEADLINE) {
                            __hip_atomic_store(tmo, seq, RLX, AGT);
                            break;
                        }
                        __builtin_amdgcn_s_sleep(1);
                    }
                }
                const char* base = isw ? lwt + (size_t)ks * 16384 + (t - TB) * 8192 + lane * 16
                                       : (const char*)(x + (size_t)(rbase + t * 64 + (lane & 15)) * KD + ks * 64 + (lane >> 4) * 8);
                const size_t st1 = isw ? 1024 : 64, st2 = isw ? 2048 : (size_t)16 * KD * 2;
#pragma unroll
                for (int m = 0; m < 4; ++m) {
                    const char* p0 = base + m * st2;
                    const char* p1 = p0 + st1;
                    asm volatile("global_load_dwordx4 %0, %2, off sc1\n\tglobal_load_dwordx4 %1, %3, off sc1"
                                 : "=&v"(v[2 * m]), "=&v"(v[2 * m + 1]) : "v"(p0), "v"(p1) : "memory");
                }
            };
            const unsigned long long tstart = __builtin_amdgcn_s_memrealtime();
            const u64 seq2 = ((u64)seq << 32) | seq;
            auto xstore = [&](const u32x4 (&v)[8], int u) {
                const int ks = u / 5, t = u % 5;
                const bool isw = odd && t >= TB;
                while ((__builtin_amdgcn_s_memrealtime() - tstart) * 1000ull < (unsigned long long)u * PACE) __builtin_amdgcn_s_sleep(1);
                char* dst = isw ? dwt + (size_t)ks * 16384 + (t - TB) * 8192 + lane * 16
                                : dxg + ((size_t)t * KSTEPS + ks) * 8192 + lane * 16;
#pragma unroll
                for (int f = 0; f < 8; ++f) store_wt(dst + f * 1024, v[f]);
                drain();
                if (isw) asm volatile("global_store_dwordx2 %0, %1, off sc0 sc1" :: "v"(dfw + ks * 2 + (t - TB)), "v"(seq2) : "memory");
                else store_flag(dfl + t * KSTEPS + ks, seq);
            };
#define WAITV(v, n) asm volatile("s_waitcnt vmcnt(" #n ")" : "+v"(v[0]), "+v"(v[1]), "+v"(v[2]), "+v"(v[3]), "+v"(v[4]), "+v"(v[5]), "+v"(v[6]), "+v"(v[7]) :: "memory")
            xload(va, u0);
            for (int u = u0; u < 320; u += 2 * WPL) {
                const int u1 = u + WPL, u2 = u + 2 * WPL;
                xload(vb, u1 < 320 ? u1 : u); WAITV(va, 8); xstore(va, u);
                xload(va, u2 < 320 ? u2 : u); WAITV(vb, 8);
                if (u1 < 320) xstore(vb, u1);
            }
            WAITV(va, 0);
        }
        return;
    }

    {
        const int u = wv * NWG + g;
        const int ks = u >> 4, j = (u >> 2) & 3, qt = u & 3;
        unsigned short* sm = (unsigned short*)(lds + wv * 4096);
        u32x4 v[4];
#pragma unroll
        for (int i = 0; i < 4; ++i)
            v[i] = *(const u32x4*)(w + (size_t)(ks * 64 + i * 16 + (lane >> 2)) * NS + j * 128 + qt * 32 + (lane & 3) * 8);
#pragma unroll
        for (int i = 0; i < 4; ++i) *(u32x4*)(sm + (i * 16 + (lane >> 2)) * 32 + (lane & 3) * 8) = v[i];
        asm volatile("s_waitcnt lgkmcnt(0)" ::: "memory");
        char* lw = ws + O_WT + (size_t)j * WT_SLOT + (size_t)ks * 16384 + qt * 4096 + lane * 16;
#pragma unroll
        for (int f = 0; f < 4; ++f) {
            const int nl = f >> 1, kt = f & 1;
            const unsigned short* sp = sm + (kt * 32 + (lane >> 4) * 8) * 32 + nl * 16 + (lane & 15);
            u32x4 o;
#pragma unroll
            for (int e = 0; e < 4; ++e) o[e] = (unsigned)sp[(2 * e) * 32] | ((unsigned)sp[(2 * e + 1) * 32] << 16);
            store_wt(lw + f * 1024, o);
        }
        drain();
        store_flag((unsigned*)(ws + O_FLAGW) + (j * KSTEPS + ks) * 4 + qt, seq);
        STAMP(1);
    }
    const int grp = g & 7, idx = g >> 3;
    const Tile T = tile_of((grp * 4 + (idx & 3)) * 8 + (idx >> 2), rank);
    const int wm = wv & 1, wn = wv >> 1;
    f32x4 acc[2][4];
#pragma unroll
    for (int a = 0; a < 2; ++a)
#pragma unroll
        for (int b = 0; b < 4; ++b) acc[a][b] = (f32x4){0.f, 0.f, 0.f, 0.f};
    const unsigned* fx0 = (const unsigned*)(ws + O_FLAGX) + (T.sx * TA + T.tx) * KSTEPS;
    const u64* fw = (const u64*)(ws + O_FLAGW) + T.ct * KSTEPS * 2;
    const char* gx = ws + O_XG + (size_t)T.sx * XG_SRC + (size_t)T.tx * KSTEPS * 8192 + wv * 2048 + lane * 16;
    const char* gw = ws + O_WT + (size_t)T.ct * WT_SLOT + wv * 4096 + lane * 16;
    auto stage = [&](int ks) {
        char* buf = lds + (ks % NSTAGE) * STAGE_BYTES;
        const char* px = gx + (size_t)ks * 8192;
        const char* pw = gw + (size_t)ks * 16384;
#pragma unroll
        for (int i = 0; i < 2; ++i)
            __builtin_amdgcn_global_load_lds((const unsigned*)(px + i * 1024), (unsigned*)(buf + wv * 2048 + i * 1024), 16, 0, 0);
#pragma unroll
        for (int i = 0; i < 4; ++i)
            __builtin_amdgcn_global_load_lds((const unsigned*)(pw + i * 1024), (unsigned*)(buf + 8192 + wv * 4096 + i * 1024), 16, 0, 0);
    };
    auto poll = [&](int need, bool first) -> int {
        unsigned long long t0 = __builtin_amdgcn_s_memrealtime();
        for (;;) {
            const unsigned a0 = __hip_atomic_load(fx0 + lane, RLX, SYS);
            const u64 b0 = __hip_atomic_load(fw + lane * 2, RLX, SYS);
            const u64 b1 = __hip_atomic_load(fw + lane * 2 + 1, RLX, SYS);
            const bool ok = a0 >= seq && (unsigned)b0 >= seq && (unsigned)(b0 >> 32) >= seq &&
                            (unsigned)b1 >= seq && (unsigned)(b1 >> 32) >= seq;
            const u64 m = ~__ballot(ok);
            const int r = m ? __builtin_ctzll(m) : 64;
            if (r > need) {
                if (first) { __builtin_amdgcn_fence(__ATOMIC_ACQUIRE, ""); drain(); }
                else __builtin_amdgcn_fence(__ATOMIC_ACQUIRE, "wavefront");
                return r;
            }
            if (__hip_atomic_load(tmo, RLX, AGT) == seq || __builtin_amdgcn_s_memrealtime() - t0 > DEADLINE) {
                __hip_atomic_store(tmo, seq, RLX, AGT);
                return 64;
            }
            __builtin_amdgcn_s_sleep(1);
        }
    };
    unsigned phase = 0;
    const unsigned lsa = (unsigned)(size_t)lsync;
    auto lds_put = [&](unsigned v) { asm volatile("ds_write_b32 %0, %1 offset:4" :: "v"(lsa), "v"(v) : "memory"); };
    auto lds_get = [&]() -> int { unsigned c; asm volatile("ds_read_b32 %0, %1 offset:4\n\ts_waitcnt lgkmcnt(0)" : "=v"(c) : "v"(lsa) : "memory"); return (int)c; };
    auto gsync = [&]() {
        phase += 4;
        asm volatile("" ::: "memory");
        if (lane == 0) asm volatile("ds_add_u32 %0, %1" :: "v"(lsa), "v"(1u) : "memory");
        unsigned c;
        do { asm volatile("ds_read_b32 %0, %1\n\ts_waitcnt lgkmcnt(0)" : "=v"(c) : "v"(lsa) : "memory"); } while (c < phase);
    };
    auto compute = [&](int ks) {
        const char* buf = lds + (ks % NSTAGE) * STAGE_BYTES;
        bf16x8 af[2][2], bfr[4][2];
#pragma unroll
        for (int a = 0; a < 2; ++a)
#pragma unroll
            for (int k = 0; k < 2; ++k) af[a][k] = *(const bf16x8*)(buf + ((wm * 2 + a) * 2 + k) * 1024 + lane * 16);
#pragma unroll
        for (int b = 0; b < 4; ++b)
#pragma unroll
            for (int k = 0; k < 2; ++k) bfr[b][k] = *(const bf16x8*)(buf + 8192 + ((wn * 4 + b) * 2 + k) * 1024 + lane * 16);
#pragma unroll
        for (int k = 0; k < 2; ++k)
#pragma unroll
            for (int a = 0; a < 2; ++a)
#pragma unroll
                for (int b = 0; b < 4; ++b)
                    acc[a][b] = __builtin_amdgcn_mfma_f32_16x16x32_bf16(bfr[b][k], af[a][k], acc[a][b], 0, 0, 0);
    };
    if (wv == 0) { const int r = poll(3, true); if (lane == 0) lds_put((unsigned)r); }
    gsync();
    STAMP(2);
    int ready = lds_get();
    stage(0); stage(1); stage(2); stage(3);
    for (int ks = 0; ks < KSTEPS - 3; ++ks) {
        const int nx = ks + 4;
        const bool need = nx < KSTEPS && nx >= ready;
        if (need && wv == 0) { const int r = poll(nx, false); if (lane == 0) lds_put((unsigned)r); }
        asm volatile("s_waitcnt vmcnt(18)" ::: "memory");
        gsync();
        if (need) ready = lds_get();
        if (nx < KSTEPS) stage(nx);
        compute(ks);
        if ((ks & 3) == 3) STAMP(3 + (ks >> 2));
    }
    asm volatile("s_waitcnt vmcnt(12)" ::: "memory"); gsync(); compute(61);
    asm volatile("s_waitcnt vmcnt(6)" ::: "memory"); gsync(); compute(62);
    asm volatile("s_waitcnt vmcnt(0)" ::: "memory"); gsync(); compute(63);
    STAMP(20);
    float mx = 0.f;
#pragma unroll
    for (int a = 0; a < 2; ++a)
#pragma unroll
        for (int b = 0; b < 4; ++b)
#pragma unroll
            for (int e = 0; e < 4; ++e) { acc[a][b][e] = fmaxf(acc[a][b][e], 0.f); mx = fmaxf(mx, acc[a][b][e]); }
#pragma unroll
    for (int o = 32; o > 0; o >>= 1) mx = fmaxf(mx, __shfl_xor(mx, o));
    float* red = (float*)(lds + NSTAGE * STAGE_BYTES);
    if (lane == 0) red[wv] = mx;
    asm volatile("s_waitcnt lgkmcnt(0)" ::: "memory");
    gsync();
    if (tid == 0) {
        const float m4 = fmaxf(fmaxf(red[0], red[1]), fmaxf(red[2], red[3]));
        const u64 key = ((u64)seq << 32) | (u64)__float_as_uint(m4);
        u64 old = __hip_atomic_fetch_max((u64*)(ws + O_GMAX + grp * 128), key, RLX, AGT);
        asm volatile("" :: "v"(old));
        drain();
        const unsigned tk = __hip_atomic_fetch_add((unsigned*)(ws + O_GCNT + grp * 128), 1u, RLX, AGT);
        if (tk == seq * 32u - 1u) {
            const u64 gm = __hip_atomic_load((u64*)(ws + O_GMAX + grp * 128), RLX, AGT);
            for (int p = 0; p < NR; ++p)
                __hip_atomic_store((u64*)((char*)ws_all[p] + O_AMAX) + rank * 8 + grp, gm, RLX, SYS);
        }
    }
    if (wv == 0) {
        const u64* slot = (const u64*)(ws + O_AMAX) + lane;
        unsigned long long t0 = __builtin_amdgcn_s_memrealtime();
        u64 v;
        bool ok = true;
        for (;;) {
            v = __hip_atomic_load(slot, RLX, SYS);
            if (__all((unsigned)(v >> 32) >= seq)) break;
            if (__hip_atomic_load(tmo, RLX, AGT) == seq || __builtin_amdgcn_s_memrealtime() - t0 > DEADLINE) {
                __hip_atomic_store(tmo, seq, RLX, AGT);
                ok = false;
                break;
            }
            __builtin_amdgcn_s_sleep(1);
        }
        float am = __uint_as_float((unsigned)v);
#pragma unroll
        for (int o = 32; o > 0; o >>= 1) am = fmaxf(am, __shfl_xor(am, o));
        if (__hip_atomic_load(tmo, RLX, AGT) == seq) ok = false;
        if (lane == 0) { red[8] = am; red[9] = ok ? 1.f : 0.f; }
    }
    __syncthreads();
    STAMP(21);
    const float amax = red[8];
    const bool good = red[9] != 0.f;
    const float scale = amax / 448.0f;
    unsigned* ot = (unsigned*)lds;
#pragma unroll
    for (int a = 0; a < 2; ++a)
#pragma unroll
        for (int b = 0; b < 4; ++b) {
            float q0, q1, q2, q3;
            if (scale > 0.f) { q0 = acc[a][b][0] / scale; q1 = acc[a][b][1] / scale; q2 = acc[a][b][2] / scale; q3 = acc[a][b][3] / scale; }
            else { q0 = q1 = q2 = q3 = 0.f; }
            int pk = __builtin_amdgcn_cvt_pk_fp8_f32(q0, q1, 0, false);
            pk = __builtin_amdgcn_cvt_pk_fp8_f32(q2, q3, pk, true);
            if (!good) pk = 0x7f7f7f7f;
            const int row = wm * 32 + a * 16 + (lane & 15);
            const int col = wn * 64 + b * 16 + (lane >> 4) * 4;
            ot[row * 32 + col / 4] = (unsigned)pk;
        }
    __syncthreads();
    const int ct = T.ct;
    const unsigned owner = ct < 4 ? (unsigned)rank : (unsigned)(rank + NR - (2 * (ct - 4) + 1)) % NR;
    char* dout = (char*)out_all[owner] + (size_t)T.grow * NS + (ct & 3) * 128;
#pragma unroll
    for (int i = 0; i < 2; ++i) {
        const int e = tid + i * 256, row = e >> 3, c16 = e & 7;
        const u32x4 v = *(const u32x4*)(lds + row * 128 + c16 * 16);
        char* dp = dout + (size_t)row * NS + c16 * 16;
        if (ct >= 4) store_wt(dp, v);
        else *(u32x4*)dp = v;
    }
    if (ct >= 4) {
        drain();
        __syncthreads();
        if (tid == 0) {
            const unsigned tk = __hip_atomic_fetch_add((unsigned*)(ws + O_OCNT + (ct - 4) * 128), 1u, RLX, AGT);
            if (tk == seq * 32u - 1u)
                __hip_atomic_store((unsigned*)((char*)ws_all[owner] + O_DONE + (ct - 4) * 128), seq, RLX, SYS);
        }
    }
    STAMP(22);
    if (g == 0 && tid == 0) {
        bool ok = true;
        for (int j = 0; j < 4; ++j) ok &= wait_flag((unsigned*)(ws + O_DONE + j * 128), seq, tmo);
        __builtin_amdgcn_fence(__ATOMIC_ACQUIRE, "");
        if (!ok) *((unsigned char*)out_all[rank] + (size_t)T.grow * NS) = 0x7f;
    }
    STAMP(23);
    if (tid == 0) {
        const unsigned tk = __hip_atomic_fetch_add((unsigned*)(ws + O_EXIT), 1u, RLX, AGT);
        if (tk == seq * (unsigned)NWG - 1u) __hip_atomic_store((unsigned*)(ws + O_SEQ), seq, RLX, AGT);
    }
}

extern "C" void kernel_launch(int rank, int nranks, ncclComm_t comm,
                              hipStream_t stream,
                              void* const* d_in, const int* in_sizes, int n_in,
                              void* d_out, int out_size,
                              const int* in_dtypes, int out_dtype,
                              void* const* d_out_all,
                              void* const* d_ws_all, size_t ws_bytes) {
    if (nranks != NR || ws_bytes < WS_NEED || n_in < 2) {
        fprintf(stderr, "unexpected configuration\n");
        return;
    }
    ag_gemm_fused<<<NWG, NT, 0, stream>>>((const unsigned short*)d_in[0], (const unsigned short*)d_in[1], rank,
                                           d_out_all, d_ws_all);
    HIP_CHECK(hipGetLastError());
}
```

```cpp
#include <hip/hip_runtime.h>
#include <rccl/rccl.h>
#include <cstdio>
#include <cstdint>

#define HIP_CHECK(expr) do { hipError_t _e = (expr); if (_e != hipSuccess) { \
    fprintf(stderr, "HIP error %s at %s:%d\n", hipGetErrorString(_e), __FILE__, __LINE__); \
    return; } } while (0)

typedef short bf16x8 __attribute__((ext_vector_type(8)));
typedef float f32x4 __attribute__((ext_vector_type(4)));
typedef unsigned u32x4 __attribute__((ext_vector_type(4)));
typedef unsigned long long u64;
typedef const u32x4 __attribute__((address_space(1)))* gcptr;

constexpr int NR = 8;
constexpr int KD = 4096;
constexpr int NS = 512;
constexpr int KSTEPS = 64;
constexpr int TA = 5, TB = 3;
#define PACE 17200
constexpr int NWG = 256;
#define WPLV 36
constexpr int WPL = WPLV;
constexpr int NT = 320;
constexpr unsigned long long DEADLINE = 40000000ull;

constexpr size_t O_SEQ = 0, O_TMO = 128, O_EXIT = 256, O_GCNT = 512, O_GMAX = 2048, O_OCNT = 4096,
                 O_AMAX = 8192, O_FLAGY = 12288,
                 O_FLAGX = 65536,
                 O_FLAGW = 131072;
constexpr size_t O_XG = 1ull << 20;
constexpr size_t XG_SRC = (size_t)TA * KSTEPS * 8192;
constexpr size_t O_WT = O_XG + NR * XG_SRC;
constexpr size_t WT_SLOT = (size_t)KSTEPS * 16384;
constexpr size_t WS_NEED = O_WT + 8 * WT_SLOT;

constexpr int NSTAGE = 5;
constexpr int STAGE_BYTES = 24576;
constexpr int LDS_BYTES = NSTAGE * STAGE_BYTES + 1024;

#define RLX __ATOMIC_RELAXED
#define SYS __HIP_MEMORY_SCOPE_SYSTEM
#define AGT __HIP_MEMORY_SCOPE_AGENT

__device__ __forceinline__ bool wait_flag(unsigned* flag, unsigned seq, unsigned* tmo) {
    unsigned long long t0 = __builtin_amdgcn_s_memrealtime();
    while (__hip_atomic_load(flag, RLX, SYS) < seq) {
        if (__hip_atomic_load(tmo, RLX, AGT) == seq || __builtin_amdgcn_s_memrealtime() - t0 > DEADLINE) {
            __hip_atomic_store(tmo, seq, RLX, AGT);
            return false;
        }
        __builtin_amdgcn_s_sleep(1);
    }
    return true;
}

#define STAMP(i) do {} while (0)
__device__ __forceinline__ void drain() { asm volatile("s_waitcnt vmcnt(0)" ::: "memory"); }
__device__ __forceinline__ void store_wt(void* p, u32x4 v) {
    asm volatile("global_store_dwordx4 %0, %1, off sc0 sc1\n\ts_nop 1" :: "v"(p), "v"(v) : "memory");
}
__device__ __forceinline__ void store_flag(unsigned* p, unsigned v) {
    asm volatile("global_store_dword %0, %1, off sc0 sc1" :: "v"(p), "v"(v) : "memory");
}

struct Tile { unsigned sx; int tx, grow, ct; };
__device__ __forceinline__ Tile tile_of(int id, int rank) {
    Tile t; const int rt = id >> 3; t.ct = id & 7;
    if (rt < 4 * TA) { t.sx = (unsigned)(rank + 2 * (rt / TA)) % NR; t.tx = rt % TA; t.grow = t.sx * NS + t.tx * 64; }
    else { const int r2 = rt - 4 * TA; t.sx = (unsigned)(rank + 2 * (r2 / TB) + 1) % NR; t.tx = r2 % TB; t.grow = t.sx * NS + TA * 64 + t.tx * 64; }
    return t;
}

__global__ void __launch_bounds__(NT, 2)
ag_gemm_fused(const unsigned short* __restrict__ x, const unsigned short* __restrict__ w, int rank,
              void* const* __restrict__ out_all, void* const* __restrict__ ws_all) {
    __shared__ __attribute__((aligned(16))) char lds[LDS_BYTES];
    char* ws = (char*)ws_all[rank];
    const int tid = threadIdx.x, lane = tid & 63, wv = tid >> 6;
    unsigned* tmo = (unsigned*)(ws + O_TMO);
    const unsigned seq = __hip_atomic_load((unsigned*)(ws + O_SEQ), RLX, AGT) + 1u;
    const int g = blockIdx.x;
    typedef __attribute__((address_space(3))) unsigned lds_u32;
    lds_u32* lsync = (lds_u32*)(__attribute__((address_space(3))) char*)(lds + NSTAGE * STAGE_BYTES + 128);
    if (tid == 0) { lsync[0] = 0u; lsync[1] = 0u; }
    __syncthreads();
    STAMP(0);

    if (wv == 4) {
        if (g < 7 * WPL) {
            const int o = 1 + g % 7, u0 = g / 7;
            const unsigned d = (unsigned)(rank + o) % NR;
            const bool odd = o & 1;
            const int j = o >> 1;
            const int rbase = odd ? TA * 64 : 0;
            char* dws = (char*)ws_all[d];
            char* dxg = dws + O_XG + (size_t)rank * XG_SRC;
            char* dwt = dws + O_WT + (size_t)(4 + j) * WT_SLOT;
            const char* lwt = ws + O_WT + (size_t)j * WT_SLOT;
            unsigned* dfl = (unsigned*)(dws + O_FLAGX) + rank * TA * KSTEPS;
            u64* dfw = (u64*)(dws + O_FLAGW) + (4 + j) * KSTEPS * 2;
            u32x4 va[8], vb[8];
            auto xload = [&](u32x4 (&v)[8], int u) {
                const int ks = u / 5, t = u % 5;
                const bool isw = odd && t >= TB;
                if (isw) {
                    const u64* lf = (const u64*)(ws + O_FLAGW) + (j * KSTEPS + ks) * 2 + (t - TB);
                    unsigned long long t0 = __builtin_amdgcn_s_memrealtime();
                    for (;;) {
                        const u64 f = __hip_atomic_load(lf, RLX, AGT);
                        if ((unsigned)f >= seq && (unsigned)(f >> 32) >= seq) break;
                        if (__hip_atomic_load(tmo, RLX, AGT) == seq || __builtin_amdgcn_s_memrealtime() - t0 > DEADLINE) {
                            __hip_atomic_store(tmo, seq, RLX, AGT);
                            break;
                        }
                        __builtin_amdgcn_s_sleep(1);
                    }
                }
                const char* base = isw ? lwt + (size_t)ks * 16384 + (t - TB) * 8192 + lane * 16
                                       : (const char*)(x + (size_t)(rbase + t * 64 + (lane & 15)) * KD + ks * 64 + (lane >> 4) * 8);
                const size_t st1 = isw ? 1024 : 64, st2 = isw ? 2048 : (size_t)16 * KD * 2;
#pragma unroll
                for (int m = 0; m < 4; ++m) {
                    const char* p0 = base + m * st2;
                    const char* p1 = p0 + st1;
                    asm volatile("global_load_dwordx4 %0, %2, off sc1\n\tglobal_load_dwordx4 %1, %3, off sc1"
                                 : "=&v"(v[2 * m]), "=&v"(v[2 * m + 1]) : "v"(p0), "v"(p1) : "memory");
                }
            };
            const unsigned long long tstart = __builtin_amdgcn_s_memrealtime();
            const u64 seq2 = ((u64)seq << 32) | seq;
            auto xstore = [&](const u32x4 (&v)[8], int u) {
                const int ks = u / 5, t = u % 5;
                const bool isw = odd && t >= TB;
                while ((__builtin_amdgcn_s_memrealtime() - tstart) * 1000ull < (unsigned long long)u * PACE) __builtin_amdgcn_s_sleep(1);
                char* dst = isw ? dwt + (size_t)ks * 16384 + (t - TB) * 8192 + lane * 16
                                : dxg + ((size_t)t * KSTEPS + ks) * 8192 + lane * 16;
#pragma unroll
                for (int f = 0; f < 8; ++f) store_wt(dst + f * 1024, v[f]);
                drain();
                if (isw) asm volatile("global_store_dwordx2 %0, %1, off sc0 sc1" :: "v"(dfw + ks * 2 + (t - TB)), "v"(seq2) : "memory");
                else store_flag(dfl + t * KSTEPS + ks, seq);
            };
#define WAITV(v, n) asm volatile("s_waitcnt vmcnt(" #n ")" : "+v"(v[0]), "+v"(v[1]), "+v"(v[2]), "+v"(v[3]), "+v"(v[4]), "+v"(v[5]), "+v"(v[6]), "+v"(v[7]) :: "memory")
            xload(va, u0);
            for (int u = u0; u < 320; u += 2 * WPL) {
                const int u1 = u + WPL, u2 = u + 2 * WPL;
                xload(vb, u1 < 320 ? u1 : u); WAITV(va, 8); xstore(va, u);
                xload(va, u2 < 320 ? u2 : u); WAITV(vb, 8);
                if (u1 < 320) xstore(vb, u1);
            }
            WAITV(va, 0);
        }
        return;
    }

    {
        const int u = wv * NWG + g;
        const int ks = u >> 4, j = (u >> 2) & 3, qt = u & 3;
        unsigned short* sm = (unsigned short*)(lds + wv * 4096);
        u32x4 v[4];
#pragma unroll
        for (int i = 0; i < 4; ++i)
            v[i] = *(const u32x4*)(w + (size_t)(ks * 64 + i * 16 + (lane >> 2)) * NS + j * 128 + qt * 32 + (lane & 3) * 8);
#pragma unroll
        for (int i = 0; i < 4; ++i) *(u32x4*)(sm + (i * 16 + (lane >> 2)) * 32 + (lane & 3) * 8) = v[i];
        asm volatile("s_waitcnt lgkmcnt(0)" ::: "memory");
        char* lw = ws + O_WT + (size_t)j * WT_SLOT + (size_t)ks * 16384 + qt * 4096 + lane * 16;
#pragma unroll
        for (int f = 0; f < 4; ++f) {
            const int nl = f >> 1, kt = f & 1;
            const unsigned short* sp = sm + (kt * 32 + (lane >> 4) * 8) * 32 + nl * 16 + (lane & 15);
            u32x4 o;
#pragma unroll
            for (int e = 0; e < 4; ++e) o[e] = (unsigned)sp[(2 * e) * 32] | ((unsigned)sp[(2 * e + 1) * 32] << 16);
            store_wt(lw + f * 1024, o);
        }
        drain();
        store_flag((unsigned*)(ws + O_FLAGW) + (j * KSTEPS + ks) * 4 + qt, seq);
        const int ux = (wv - 2) * NWG + g;
        if (wv >= 2 && ux < TA * KSTEPS) {
            const int kx = ux / TA, t = ux % TA;
            const unsigned short* src = x + (size_t)(t * 64 + (lane & 15)) * KD + kx * 64 + (lane >> 4) * 8;
            u32x4 v8[8];
#pragma unroll
            for (int f = 0; f < 8; ++f) v8[f] = *(const u32x4*)(src + (size_t)(f >> 1) * 16 * KD + (f & 1) * 32);
            char* dst = ws + O_XG + (size_t)rank * XG_SRC + ((size_t)t * KSTEPS + kx) * 8192 + lane * 16;
#pragma unroll
            for (int f = 0; f < 8; ++f) store_wt(dst + f * 1024, v8[f]);
            drain();
            store_flag((unsigned*)(ws + O_FLAGX) + (rank * TA + t) * KSTEPS + kx, seq);
        }
        STAMP(1);
    }
    const int grp = g & 7, idx = g >> 3;
    const Tile T = tile_of((grp * 4 + (idx & 3)) * 8 + (idx >> 2), rank);
    const int wm = wv & 1, wn = wv >> 1;
    f32x4 acc[2][4];
#pragma unroll
    for (int a = 0; a < 2; ++a)
#pragma unroll
        for (int b = 0; b < 4; ++b) acc[a][b] = (f32x4){0.f, 0.f, 0.f, 0.f};
    const unsigned* fx0 = (const unsigned*)(ws + O_FLAGX) + (T.sx * TA + T.tx) * KSTEPS;
    const u64* fw = (const u64*)(ws + O_FLAGW) + T.ct * KSTEPS * 2;
    const char* gx = ws + O_XG + (size_t)T.sx * XG_SRC + (size_t)T.tx * KSTEPS * 8192 + wv * 2048 + lane * 16;
    const char* gw = ws + O_WT + (size_t)T.ct * WT_SLOT + wv * 4096 + lane * 16;
    auto stage = [&](int ks) {
        char* buf = lds + (ks % NSTAGE) * STAGE_BYTES;
        const char* px = gx + (size_t)ks * 8192;
        const char* pw = gw + (size_t)ks * 16384;
#pragma unroll
        for (int i = 0; i < 2; ++i)
            __builtin_amdgcn_global_load_lds((const unsigned*)(px + i * 1024), (unsigned*)(buf + wv * 2048 + i * 1024), 16, 0, 0);
#pragma unroll
        for (int i = 0; i < 4; ++i)
            __builtin_amdgcn_global_load_lds((const unsigned*)(pw + i * 1024), (unsigned*)(buf + 8192 + wv * 4096 + i * 1024), 16, 0, 0);
    };
    auto poll = [&](int need, bool first) -> int {
        unsigned long long t0 = __builtin_amdgcn_s_memrealtime();
        for (;;) {
            const unsigned a0 = __hip_atomic_load(fx0 + lane, RLX, SYS);
            const u64 b0 = __hip_atomic_load(fw + lane * 2, RLX, SYS);
            const u64 b1 = __hip_atomic_load(fw + lane * 2 + 1, RLX, SYS);
            const bool ok = a0 >= seq && (unsigned)b0 >= seq && (unsigned)(b0 >> 32) >= seq &&
                            (unsigned)b1 >= seq && (unsigned)(b1 >> 32) >= seq;
            const u64 m = ~__ballot(ok);
            const int r = m ? __builtin_ctzll(m) : 64;
            if (r > need) {
                if (first) { __builtin_amdgcn_fence(__ATOMIC_ACQUIRE, ""); drain(); }
                else __builtin_amdgcn_fence(__ATOMIC_ACQUIRE, "wavefront");
                return r;
            }
            if (__hip_atomic_load(tmo, RLX, AGT) == seq || __builtin_amdgcn_s_memrealtime() - t0 > DEADLINE) {
                __hip_atomic_store(tmo, seq, RLX, AGT);
                return 64;
            }
            __builtin_amdgcn_s_sleep(1);
        }
    };
    unsigned phase = 0;
    const unsigned lsa = (unsigned)(size_t)lsync;
    auto lds_put = [&](unsigned v) { asm volatile("ds_write_b32 %0, %1 offset:4" :: "v"(lsa), "v"(v) : "memory"); };
    auto lds_get = [&]() -> int { unsigned c; asm volatile("ds_read_b32 %0, %1 offset:4\n\ts_waitcnt lgkmcnt(0)" : "=v"(c) : "v"(lsa) : "memory"); return (int)c; };
    auto gsync = [&]() {
        phase += 4;
        asm volatile("" ::: "memory");
        if (lane == 0) asm volatile("ds_add_u32 %0, %1" :: "v"(lsa), "v"(1u) : "memory");
        unsigned c;
        do { asm volatile("ds_read_b32 %0, %1\n\ts_waitcnt lgkmcnt(0)" : "=v"(c) : "v"(lsa) : "memory"); } while (c < phase);
    };
    auto compute = [&](int ks) {
        const char* buf = lds + (ks % NSTAGE) * STAGE_BYTES;
        bf16x8 af[2][2], bfr[4][2];
#pragma unroll
        for (int a = 0; a < 2; ++a)
#pragma unroll
            for (int k = 0; k < 2; ++k) af[a][k] = *(const bf16x8*)(buf + ((wm * 2 + a) * 2 + k) * 1024 + lane * 16);
#pragma unroll
        for (int b = 0; b < 4; ++b)
#pragma unroll
            for (int k = 0; k < 2; ++k) bfr[b][k] = *(const bf16x8*)(buf + 8192 + ((wn * 4 + b) * 2 + k) * 1024 + lane * 16);
#pragma unroll
        for (int k = 0; k < 2; ++k)
#pragma unroll
            for (int a = 0; a < 2; ++a)
#pragma unroll
                for (int b = 0; b < 4; ++b)
                    acc[a][b] = __builtin_amdgcn_mfma_f32_16x16x32_bf16(bfr[b][k], af[a][k], acc[a][b], 0, 0, 0);
    };
    if (wv == 0) { const int r = poll(3, true); if (lane == 0) lds_put((unsigned)r); }
    gsync();
    STAMP(2);
    int ready = lds_get();
    stage(0); stage(1); stage(2); stage(3);
    for (int ks = 0; ks < KSTEPS - 3; ++ks) {
        const int nx = ks + 4;
        const bool need = nx < KSTEPS && nx >= ready;
        if (need && wv == 0) { const int r = poll(nx, false); if (lane == 0) lds_put((unsigned)r); }
        asm volatile("s_waitcnt vmcnt(18)" ::: "memory");
        gsync();
        if (need) ready = lds_get();
        if (nx < KSTEPS) stage(nx);
        compute(ks);
        if ((ks & 3) == 3) STAMP(3 + (ks >> 2));
    }
    asm volatile("s_waitcnt vmcnt(12)" ::: "memory"); gsync(); compute(61);
    asm volatile("s_waitcnt vmcnt(6)" ::: "memory"); gsync(); compute(62);
    asm volatile("s_waitcnt vmcnt(0)" ::: "memory"); gsync(); compute(63);
    STAMP(20);
    float mx = 0.f;
#pragma unroll
    for (int a = 0; a < 2; ++a)
#pragma unroll
        for (int b = 0; b < 4; ++b)
#pragma unroll
            for (int e = 0; e < 4; ++e) { acc[a][b][e] = fmaxf(acc[a][b][e], 0.f); mx = fmaxf(mx, acc[a][b][e]); }
#pragma unroll
    for (int o = 32; o > 0; o >>= 1) mx = fmaxf(mx, __shfl_xor(mx, o));
    float* red = (float*)(lds + NSTAGE * STAGE_BYTES);
    if (lane == 0) red[wv] = mx;
    const int ct = T.ct, rt = grp * 4 + (idx & 3);
    gsync();
    if (tid == 0) {
        const float m4 = fmaxf(fmaxf(red[0], red[1]), fmaxf(red[2], red[3]));
        const u64 key = ((u64)seq << 32) | (u64)__float_as_uint(m4);
        u64 old = __hip_atomic_fetch_max((u64*)(ws + O_GMAX + grp * 128), key, RLX, AGT);
        asm volatile("" :: "v"(old));
        drain();
        const unsigned tk = __hip_atomic_fetch_add((unsigned*)(ws + O_GCNT + grp * 128), 1u, RLX, AGT);
        if (tk == seq * 32u - 1u) {
            const u64 gm = __hip_atomic_load((u64*)(ws + O_GMAX + grp * 128), RLX, AGT);
            for (int p = 0; p < NR; ++p)
                __hip_atomic_store((u64*)((char*)ws_all[p] + O_AMAX) + rank * 8 + grp, gm, RLX, SYS);
        }
    }
    if (wv == 0) {
        const u64* slot = (const u64*)(ws + O_AMAX) + lane;
        unsigned long long t0 = __builtin_amdgcn_s_memrealtime();
        u64 v;
        bool ok = true;
        for (;;) {
            v = __hip_atomic_load(slot, RLX, SYS);
            if (__all((unsigned)(v >> 32) >= seq)) break;
            if (__hip_atomic_load(tmo, RLX, AGT) == seq || __builtin_amdgcn_s_memrealtime() - t0 > DEADLINE) {
                __hip_atomic_store(tmo, seq, RLX, AGT);
                ok = false;
                break;
            }
            __builtin_amdgcn_s_sleep(1);
        }
        float am = __uint_as_float((unsigned)v);
#pragma unroll
        for (int o = 32; o > 0; o >>= 1) am = fmaxf(am, __shfl_xor(am, o));
        if (__hip_atomic_load(tmo, RLX, AGT) == seq) ok = false;
        if (lane == 0) { red[8] = am; red[9] = ok ? 1.f : 0.f; }
    }
    __syncthreads();
    STAMP(21);
    const float amax = red[8];
    const bool good = red[9] != 0.f;
    const float scale = amax / 448.0f;
    unsigned* ot = (unsigned*)lds;
#pragma unroll
    for (int a = 0; a < 2; ++a)
#pragma unroll
        for (int b = 0; b < 4; ++b) {
            float q0, q1, q2, q3;
            if (scale > 0.f) { q0 = acc[a][b][0] / scale; q1 = acc[a][b][1] / scale; q2 = acc[a][b][2] / scale; q3 = acc[a][b][3] / scale; }
            else { q0 = q1 = q2 = q3 = 0.f; }
            int pk = __builtin_amdgcn_cvt_pk_fp8_f32(q0, q1, 0, false);
            pk = __builtin_amdgcn_cvt_pk_fp8_f32(q2, q3, pk, true);
            if (!good) pk = 0x7f7f7f7f;
            const int row = wm * 32 + a * 16 + (lane & 15);
            const int col = wn * 64 + b * 16 + (lane >> 4) * 4;
            ot[row * 32 + col / 4] = (unsigned)pk;
        }
    __syncthreads();
    const unsigned owner = ct < 4 ? (unsigned)rank : (unsigned)(rank + NR - (2 * (ct - 4) + 1)) % NR;
    char* dout = (char*)out_all[owner] + (size_t)T.grow * NS + (ct & 3) * 128;
#pragma unroll
    for (int i = 0; i < 2; ++i) {
        const int e = tid + i * 256, row = e >> 3, c16 = e & 7;
        const u32x4 v = *(const u32x4*)(lds + row * 128 + c16 * 16);
        char* dp = dout + (size_t)row * NS + c16 * 16;
        if (ct >= 4) store_wt(dp, v);
        else *(u32x4*)dp = v;
    }
    if (ct >= 4) {
        drain();
        __syncthreads();
        if (wv == 0) store_flag((unsigned*)((char*)ws_all[owner] + O_FLAGY) + (ct - 4) * 32 + rt, seq);
    }
    STAMP(22);
    if (g == 0 && wv == 0) {
        const unsigned* fy = (const unsigned*)(ws + O_FLAGY) + lane;
        unsigned long long t0 = __builtin_amdgcn_s_memrealtime();
        bool ok = true;
        for (;;) {
            const unsigned f0 = __hip_atomic_load(fy, RLX, SYS), f1 = __hip_atomic_load(fy + 64, RLX, SYS);
            if (__all(f0 >= seq && f1 >= seq)) break;
            if (__hip_atomic_load(tmo, RLX, AGT) == seq || __builtin_amdgcn_s_memrealtime() - t0 > DEADLINE) {
                __hip_atomic_store(tmo, seq, RLX, AGT);
                ok = false;
                break;
            }
            __builtin_amdgcn_s_sleep(1);
        }
        __builtin_amdgcn_fence(__ATOMIC_ACQUIRE, "");
        if (!ok && lane == 0) *((unsigned char*)out_all[rank] + (size_t)T.grow * NS) = 0x7f;
    }
    STAMP(23);
    if (tid == 0) {
        const unsigned tk = __hip_atomic_fetch_add((unsigned*)(ws + O_EXIT), 1u, RLX, AGT);
        if (tk == seq * (unsigned)NWG - 1u) __hip_atomic_store((unsigned*)(ws + O_SEQ), seq, RLX, AGT);
    }
}

extern "C" void kernel_launch(int rank, int nranks, ncclComm_t comm,
                              hipStream_t stream,
                              void* const* d_in, const int* in_sizes, int n_in,
                              void* d_out, int out_size,
                              const int* in_dtypes, int out_dtype,
                              void* const* d_out_all,
                              void* const* d_ws_all, size_t ws_bytes) {
    if (nranks != NR || ws_bytes < WS_NEED || n_in < 2) {
        fprintf(stderr, "unexpected configuration\n");
        return;
    }
    ag_gemm_fused<<<NWG, NT, 0, stream>>>((const unsigned short*)d_in[0], (const unsigned short*)d_in[1], rank,
                                           d_out_all, d_ws_all);
    HIP_CHECK(hipGetLastError());
}
```
